```python
import numpy as np

kernel_name = "gpu_ag_gemm_m4096_k4096_n16384_e4m3e5m2_dqrelu"


ABSMAX_RTOL = 0.05

COMPUTE_DTYPE = "float32"
EPILOGUE = "dequant_relu"
IN_QUANT = None
OUT_QUANT = None
IN_FP8 = "e4m3e5m2"
IN_DTYPES = {"x": "e4m3", "w_mat": "e5m2",
             "scale_x": "float32", "scale_w": "float32"}
WORLD_SIZE = 8

def _snap_fp8(x, mant_bits, max_abs, min_normal):
    a = np.abs(x.astype(np.float64))
    _m, _e = np.frexp(a)
    q = np.where(a >= min_normal, np.ldexp(1.0, _e - 1 - mant_bits),
                 min_normal * 2.0 ** -mant_bits)
    snapped = np.minimum(np.round(a / q) * q, max_abs)
    return (np.sign(x) * snapped).astype(np.float32)

def setup_inputs():

    x = _snap_fp8(np.random.randn(4096, 4096).astype(np.float32), 3, 448.0, 2.0 ** -6)
    w_mat = _snap_fp8(np.random.randn(4096, 16384).astype(np.float32), 2, 57344.0, 2.0 ** -14)
    scale_x = (0.02 * (1.0 + np.random.rand(1))).astype(np.float32)
    scale_w = (0.02 * (1.0 + np.random.rand(1))).astype(np.float32)
    return {"x": x, "w_mat": w_mat, "scale_x": scale_x, "scale_w": scale_w}

def reference(x, w_mat, scale_x, scale_w):
    acc = (x @ w_mat).astype(np.float32)
    y = (acc * (scale_x * scale_w)).astype(np.float32)
    return np.maximum(y, 0.0).astype(np.float32)
```

```cpp
#include <hip/hip_runtime.h>
#include <rccl/rccl.h>
#include <cstdio>
#include <cstdint>

#define HIP_CHECK(expr) do { hipError_t _e = (expr); if (_e != hipSuccess) { \
    fprintf(stderr, "HIP error %s at %s:%d\n", hipGetErrorString(_e), __FILE__, __LINE__); \
    return; } } while (0)

typedef int v8i __attribute__((ext_vector_type(8)));
typedef int v4i __attribute__((ext_vector_type(4)));
typedef int v2i __attribute__((ext_vector_type(2)));
typedef float v16f __attribute__((ext_vector_type(16)));

#define NFAST 2
namespace {

constexpr int NR = 8;
constexpr int M_LOC = 512;
constexpr int KDIM = 4096;
constexpr int N_LOC = 2048;
constexpr int BM = 128, BN = 256, BK = 64;
constexpr int NKS = KDIM / BK;
constexpr int NWG = 256;
constexpr int NTHREADS = 512;
constexpr int NSTAGE = 4;
constexpr int A_STAGE = BM * BK;
constexpr int B_STAGE = BK * BN;
constexpr int STAGE_BYTES = A_STAGE + B_STAGE;

constexpr size_t WS_TMO = 128;
constexpr size_t WS_FLAGS = 4096;
constexpr size_t WS_TICKET = 65536;
constexpr size_t WS_STAGE = 1u << 20;
constexpr size_t SHARD_BYTES = (size_t)M_LOC * KDIM;
constexpr size_t WS_NEEDED = WS_STAGE + 2 * SHARD_BYTES;
constexpr unsigned long long DEADLINE = 20000000ull * 5;

struct Params {
  const uint8_t* x;
  const uint8_t* w;
  const float* sx;
  const float* sw;
  float* out;
  char* ws[NR];
  unsigned srcs;
  int rank;
};

__device__ __forceinline__ v4i gload16(const void* p) {
  v4i r;
  asm volatile("global_load_dwordx4 %0, %1, off" : "=v"(r) : "v"(p) : "memory");
  return r;
}
__device__ __forceinline__ void gstore16_sys(void* p, v4i v) {
  asm volatile("global_store_dwordx4 %0, %1, off sc0 sc1\n\ts_nop 1" ::"v"(p), "v"(v) : "memory");
}
__device__ __forceinline__ void gstore4_sys(void* p, unsigned v) {
  asm volatile("global_store_dword %0, %1, off sc0 sc1" ::"v"(p), "v"(v) : "memory");
}
#define WAIT_VM(n) asm volatile("s_waitcnt vmcnt(%0)" :: "n"(n) : "memory")

__device__ __forceinline__ bool wait_flags(const unsigned* f, unsigned* tmo, unsigned call) {
  unsigned long long t0 = __builtin_amdgcn_s_memrealtime();
  for (unsigned it = 1;; ++it) {
    unsigned v = __hip_atomic_load(f, __ATOMIC_RELAXED, __HIP_MEMORY_SCOPE_SYSTEM);
    if (__all((int)(v - call) >= 0)) return true;
    if ((it & 31u) == 0) {
      unsigned t = __hip_atomic_load(tmo, __ATOMIC_RELAXED, __HIP_MEMORY_SCOPE_AGENT);
      if (t == call || __builtin_amdgcn_s_memrealtime() - t0 > DEADLINE) {
        __hip_atomic_store(tmo, call, __ATOMIC_RELAXED, __HIP_MEMORY_SCOPE_AGENT);
        return false;
      }
      __builtin_amdgcn_s_sleep(2);
    }
  }
}

#define DS_TR8(dst, addr, off) asm volatile("ds_read_b64_tr_b8 %0, %1 offset:%2" : "=v"(dst) : "v"(addr), "i"(off))
#define DS_B128(dst, addr, off) asm volatile("ds_read_b128 %0, %1 offset:%2" : "=v"(dst) : "v"(addr), "i"(off))

struct Frags {
  v4i a[2][2];
  v2i b[2][4];
};

__device__ __forceinline__ void read_frags(Frags& F, unsigned aaddr, unsigned b0addr, unsigned b1addr) {
  constexpr int OA = 0, OB = A_STAGE;
  DS_TR8(F.b[0][0], b0addr, OB + 0);    DS_TR8(F.b[0][1], b0addr, OB + 2048);
  DS_TR8(F.b[0][2], b0addr, OB + 4096); DS_TR8(F.b[0][3], b0addr, OB + 6144);
  DS_B128(F.a[0][0], aaddr, OA + 0);    DS_B128(F.a[0][1], aaddr, OA + 1024);
  DS_TR8(F.b[1][0], b1addr, OB + 0);    DS_TR8(F.b[1][1], b1addr, OB + 2048);
  DS_TR8(F.b[1][2], b1addr, OB + 4096); DS_TR8(F.b[1][3], b1addr, OB + 6144);
  DS_B128(F.a[1][0], aaddr, OA + 2048); DS_B128(F.a[1][1], aaddr, OA + 3072);
}
#define MFMA(acc, a, b) acc = __builtin_amdgcn_mfma_scale_f32_32x32x64_f8f6f4(a, b, acc, 0, 1, 0, 0x7F7F7F7F, 0, 0x7F7F7F7F)
#define CAT4(x, y) __builtin_shufflevector(x, y, 0, 1, 2, 3, 4, 5, 6, 7)
#define CAT2(x, y) __builtin_shufflevector(x, y, 0, 1, 2, 3)
__device__ __forceinline__ void mma_frags(Frags& F, v16f (&acc)[2][2]) {
  asm volatile("s_waitcnt lgkmcnt(6)" : "+v"(F.b[0][0]), "+v"(F.b[0][1]), "+v"(F.b[0][2]), "+v"(F.b[0][3]), "+v"(F.a[0][0]), "+v"(F.a[0][1]));
  v8i b0 = CAT4(CAT2(F.b[0][0], F.b[0][1]), CAT2(F.b[0][2], F.b[0][3]));
  v8i a0 = CAT4(F.a[0][0], F.a[0][1]);
  MFMA(acc[0][0], a0, b0);
  __builtin_amdgcn_sched_barrier(0);
  asm volatile("s_waitcnt lgkmcnt(2)" : "+v"(F.b[1][0]), "+v"(F.b[1][1]), "+v"(F.b[1][2]), "+v"(F.b[1][3]));
  v8i b1 = CAT4(CAT2(F.b[1][0], F.b[1][1]), CAT2(F.b[1][2], F.b[1][3]));
  MFMA(acc[0][1], a0, b1);
  __builtin_amdgcn_sched_barrier(0);
  asm volatile("s_waitcnt lgkmcnt(0)" : "+v"(F.a[1][0]), "+v"(F.a[1][1]));
  v8i a1 = CAT4(F.a[1][0], F.a[1][1]);
  MFMA(acc[1][0], a1, b0); MFMA(acc[1][1], a1, b1);
  asm volatile("" : "+v"(acc[0][0]), "+v"(acc[0][1]), "+v"(acc[1][0]), "+v"(acc[1][1]));
}

__global__ __launch_bounds__(NTHREADS, 2) void ag_gemm_kernel(Params p) {
  __shared__ __attribute__((aligned(16))) char smem[NSTAGE * STAGE_BYTES + 16];
  const int tid = threadIdx.x;
  const int lane = tid & 63;
  const int wv = __builtin_amdgcn_readfirstlane(tid >> 6);
  char* myws = p.ws[0];
  const int bid = __builtin_amdgcn_readfirstlane(blockIdx.x);
#define STAMP(i)

  const int srcsel = bid & 7;
  const int idx = bid >> 3;
  const int rb = idx >> 3, cb = idx & 7;
  int src = (p.srcs >> (4 * srcsel)) & 7;
  const char* srcws = p.ws[srcsel];
  const uint8_t* wptr = p.w;
  float* outp = p.out;
  char* peerws = p.ws[lane & 7];
  int rank = p.rank;
  asm volatile("" : "+s"(srcws), "+s"(wptr), "+s"(outp), "+s"(myws), "+s"(rank), "+s"(src));

  const int g32 = bid & 15, kc = bid >> 4;
  v4i xd = gload16(p.x + (size_t)(g32 * 32 + (lane & 31)) * KDIM + kc * 256 + (wv >> 1) * 64 + (lane >> 5) * 32 + (wv & 1) * 16);
  float sxv, swv;
  asm volatile("global_load_dword %0, %1, off" : "=v"(sxv) : "v"(p.sx) : "memory");
  asm volatile("global_load_dword %0, %1, off" : "=v"(swv) : "v"(p.sw) : "memory");

  unsigned* s_call = (unsigned*)(smem + NSTAGE * STAGE_BYTES);
  if (tid == 0) {
    unsigned t = __hip_atomic_fetch_add((unsigned*)(myws + WS_TICKET) + bid * 16, 1u, __ATOMIC_RELAXED, __HIP_MEMORY_SCOPE_AGENT);
    s_call[0] = t + 1u;
  }
  asm volatile("s_waitcnt vmcnt(0) lgkmcnt(0)" : "+v"(xd), "+v"(sxv), "+v"(swv), "+v"(peerws) :: "memory");
  const float sc = sxv * swv;
  __syncthreads();
  const unsigned call = __builtin_amdgcn_readfirstlane(s_call[0]);
  const unsigned parity = call & 1u;
  STAMP(5);
  {
    char* d = myws + WS_STAGE + (size_t)parity * SHARD_BYTES +
              (size_t)((((g32 >> 2) * NKS + kc * 4 + (wv >> 1)) * 4 + (g32 & 3)) * 2 + (wv & 1)) * 1024 + lane * 16;
    gstore16_sys(d, xd);
  }
  WAIT_VM(0);
  __syncthreads();

  unsigned* tmo = (unsigned*)(myws + WS_TMO);
  if (wv == 0) {
    if (lane < NR) gstore4_sys((unsigned*)(peerws + WS_FLAGS) + rank * 256 + g32 * 16 + kc, call);
    STAMP(6);
    bool ok = wait_flags((const unsigned*)(myws + WS_FLAGS) + src * 256 + rb * 64 + lane, tmo, call);
    if (lane == 0) s_call[1] = ok ? 0u : 1u;
  }

  const int wm = wv >> 2, wn = wv & 3;
  const char* Abase = srcws + WS_STAGE + (size_t)parity * SHARD_BYTES + (size_t)rb * NKS * 8192;
  __amdgpu_buffer_rsrc_t rsB = __builtin_amdgcn_make_buffer_rsrc((void*)(wptr + cb * BN), 0, KDIM * N_LOC - cb * BN, 0x00020000);
  __amdgpu_buffer_rsrc_t rsO = __builtin_amdgcn_make_buffer_rsrc((void*)(outp + (size_t)(src * M_LOC + rb * BM) * N_LOC + cb * BN), 0,
                                                                  BM * N_LOC * 4 - cb * BN * 4, 0x00020000);
  const int voffA = wv * 1024 + lane * 16;
  const int kq = lane >> 4;
  int voffB[2];
#pragma unroll
  for (int jp = 0; jp < 2; ++jp)
    voffB[jp] = (wv * 8 + jp * 4 + kq) * N_LOC + (((lane & 15) ^ ((jp * 4 + kq) << 1)) * 16);
  const int voffO = ((wm * 64 + 4 * (lane >> 5)) * N_LOC + wn * 64 + (lane & 31)) * 4;

  v16f acc[2][2];
#pragma unroll
  for (int f = 0; f < 2; ++f)
#pragma unroll
    for (int n = 0; n < 2; ++n)
#pragma unroll
      for (int e = 0; e < 16; ++e) acc[f][n][e] = 0.f;

  const unsigned lds0 = (unsigned)(size_t)(__attribute__((address_space(3))) char*)smem;
  const unsigned a_off = lds0 + wm * 4096 + lane * 16;
  const int gq = lane >> 4, hq = gq >> 1, cs = gq & 1, q = (lane & 15) >> 1, pp = lane & 1;
  unsigned b_off[2];
#pragma unroll
  for (int n = 0; n < 2; ++n) {
    int j = wn * 4 + n * 2 + cs;
    b_off[n] = lds0 + (32 * hq + q) * 256 + ((j ^ (q << 1)) * 16) + 8 * pp;
  }

#define BLDS(rs, lptr, voff, soff) __builtin_amdgcn_raw_ptr_buffer_load_lds(rs, (__attribute__((address_space(3))) void*)(lptr), 16, voff, soff, 0, 0)
#define ISSUE(s, buf) do { char* sb = smem + (buf) * STAGE_BYTES; const int s_ = (s) < NKS ? (s) : NKS - 1; \
    BLDS(rsA, sb + wv * 1024, voffA, s_ * 8192); \
    BLDS(rsB, sb + A_STAGE + (wv * 2 + 0) * 1024, voffB[0], s_ * BK * N_LOC); \
    BLDS(rsB, sb + A_STAGE + (wv * 2 + 1) * 1024, voffB[1], s_ * BK * N_LOC); \
  } while (0)
#define ITER(s, BUF, W) do { \
    WAIT_VM(3 * ((W) - 1)); \
    __builtin_amdgcn_s_barrier(); \
    Frags F; \
    read_frags(F, a_off + (BUF) * STAGE_BYTES, b_off[0] + (BUF) * STAGE_BYTES, b_off[1] + (BUF) * STAGE_BYTES); \
    ISSUE((s) + (W), ((BUF) + (W)) % NSTAGE); \
    __builtin_amdgcn_sched_barrier(0); \
    mma_frags(F, acc); \
    __builtin_amdgcn_sched_barrier(0); \
  } while (0)

  __syncthreads();
  const bool bad = __builtin_amdgcn_readfirstlane(s_call[1]) != 0;
  STAMP(1);
  __amdgpu_buffer_rsrc_t rsA = __builtin_amdgcn_make_buffer_rsrc((void*)Abase, 0, bad ? 0 : NKS * 8192, 0x00020000);
  if (rb < NFAST) {
    ISSUE(0, 0); ISSUE(1, 1); ISSUE(2, 2);
#pragma nounroll
    for (int s0 = 0; s0 < NKS; s0 += NSTAGE) {
#pragma unroll
      for (int b = 0; b < NSTAGE; ++b) ITER(s0 + b, b, 3);
    }
  } else {
    ISSUE(0, 0); ISSUE(1, 1);
#pragma nounroll
    for (int s0 = 0; s0 < NKS; s0 += NSTAGE) {
#pragma unroll
      for (int b = 0; b < NSTAGE; ++b) ITER(s0 + b, b, 2);
    }
  }
  STAMP(2);

#pragma unroll
  for (int f = 0; f < 2; ++f)
#pragma unroll
    for (int n = 0; n < 2; ++n)
#pragma unroll
      for (int e = 0; e < 16; ++e) {
        float y = fmaxf(acc[f][n][e] * sc, 0.f);
        if (bad) y = __builtin_nanf("");
        __builtin_amdgcn_raw_buffer_store_b32(__float_as_uint(y), rsO, voffO + (f * 32 + (e & 3) + 8 * (e >> 2)) * N_LOC * 4 + n * 128, 0, 2);
      }
}

}

extern "C" void kernel_launch(int rank, int nranks, ncclComm_t comm,
                              hipStream_t stream,
                              void* const* d_in, const int* in_sizes, int n_in,
                              void* d_out, int out_size,
                              const int* in_dtypes, int out_dtype,
                              void* const* d_out_all,
                              void* const* d_ws_all, size_t ws_bytes) {
  if (nranks != NR || ws_bytes < WS_NEEDED || rank < 0 || rank >= NR) {
    fprintf(stderr, "unexpected configuration\n");
    return;
  }
  Params p;
  p.x = (const uint8_t*)d_in[0];
  p.w = (const uint8_t*)d_in[1];
  p.sx = (const float*)d_in[2];
  p.sw = (const float*)d_in[3];
  p.out = (float*)d_out;
#define HALF_MASK 0x4Du
  int order[NR], n = 0;
  order[n++] = rank;
  for (int pass = 0; pass < 2; ++pass)
    for (int i = 1; i < NR; ++i) {
      const int q = (rank + i) % NR;
      const bool same = ((HALF_MASK >> q) & 1u) == ((HALF_MASK >> rank) & 1u);
      if (same == (pass == 0)) order[n++] = q;
    }
  p.srcs = 0;
  for (int i = 0; i < NR; ++i) {
    p.ws[i] = (char*)d_ws_all[order[i]];
    p.srcs |= (unsigned)order[i] << (4 * i);
  }
  p.rank = rank;
  ag_gemm_kernel<<<NWG, NTHREADS, 0, stream>>>(p);
  HIP_CHECK(hipGetLastError());
}
```
